```python
import numpy as np

kernel_name = "gpu_gemm_a2a_m4096_k4096_n4096_e4m3e5m2_dqsilu"


ABSMAX_RTOL = 0.05

COMPUTE_DTYPE = "float32"
EPILOGUE = "dequant_silu"
IN_QUANT = None
OUT_QUANT = None
IN_FP8 = "e4m3e5m2"
IN_DTYPES = {"x": "e4m3", "w_mat": "e5m2",
             "scale_x": "float32", "scale_w": "float32"}
WORLD_SIZE = 8

def _snap_fp8(x, mant_bits, max_abs, min_normal):
    a = np.abs(x.astype(np.float64))
    _m, _e = np.frexp(a)
    q = np.where(a >= min_normal, np.ldexp(1.0, _e - 1 - mant_bits),
                 min_normal * 2.0 ** -mant_bits)
    snapped = np.minimum(np.round(a / q) * q, max_abs)
    return (np.sign(x) * snapped).astype(np.float32)

def setup_inputs():

    x = _snap_fp8(np.random.randn(4096, 4096).astype(np.float32), 3, 448.0, 2.0 ** -6)
    w_mat = _snap_fp8(np.random.randn(4096, 4096).astype(np.float32), 2, 57344.0, 2.0 ** -14)
    scale_x = (0.02 * (1.0 + np.random.rand(1))).astype(np.float32)
    scale_w = (0.02 * (1.0 + np.random.rand(1))).astype(np.float32)
    return {"x": x, "w_mat": w_mat, "scale_x": scale_x, "scale_w": scale_w}

def reference(x, w_mat, scale_x, scale_w):
    acc = (x @ w_mat).astype(np.float32)
    y = (acc * (scale_x * scale_w)).astype(np.float32)
    return (y / (1.0 + np.exp(-np.clip(y, -60.0, 60.0)))).astype(np.float32)
```

```cpp
#include <hip/hip_runtime.h>
#include <rccl/rccl.h>
#include <stdint.h>
#include <cstdio>

#define HIP_CHECK(expr) do { hipError_t _e = (expr); if (_e != hipSuccess) { \
    fprintf(stderr, "HIP error %s at %s:%d\n", hipGetErrorString(_e), __FILE__, __LINE__); \
    return; } } while (0)

namespace {

constexpr int NRANKS = 8;
constexpr int M_LOC  = 512;
constexpr int K_DIM  = 4096;
constexpr int N_DIM  = 4096;
constexpr int N_LOC  = 512;

constexpr int BM = 64, BN = 128, BK = 128;
constexpr int NSTAGE = 4;
constexpr int A_BYTES = BM * BK;
constexpr int B_BYTES = BK * BN;
constexpr int STAGE_BYTES = A_BYTES + B_BYTES;
constexpr int NT = K_DIM / BK;
constexpr int NTHREADS = 256;
constexpr int GRID = (M_LOC / BM) * (N_DIM / BN);

constexpr int W_CALLS  = 0;
constexpr int W_TICKET = 32;
constexpr int W_TMO    = 64;
constexpr int W_READY  = 256;
constexpr int W_DONE   = 512;
constexpr unsigned long long DEADLINE_TICKS = 100000000ull;

typedef int   v2i __attribute__((ext_vector_type(2)));
typedef int   v4i __attribute__((ext_vector_type(4)));
typedef int   v8i __attribute__((ext_vector_type(8)));
typedef float v4f __attribute__((ext_vector_type(4)));

struct Params {
  const uint8_t* x;
  const uint8_t* w;
  const float* sx;
  const float* sw;
  float* out[NRANKS];
  uint32_t* ws[NRANKS];
  int rank;
};

#define STAMP(i) do {} while (0)
#define WAIT_VM(n) asm volatile("s_waitcnt vmcnt(" #n ")" ::: "memory")

__device__ __forceinline__ uint32_t ld_sys(const uint32_t* p) {
  return __hip_atomic_load(p, __ATOMIC_RELAXED, __HIP_MEMORY_SCOPE_SYSTEM);
}
__device__ __forceinline__ uint32_t ld_agent(const uint32_t* p) {
  return __hip_atomic_load(p, __ATOMIC_RELAXED, __HIP_MEMORY_SCOPE_AGENT);
}

__device__ __forceinline__ void glds16(const uint8_t* src, uint32_t lds_uniform) {
  asm volatile("s_mov_b32 m0, %1\n\ts_nop 0\n\tglobal_load_lds_dwordx4 %0, off"
               :: "v"(src), "s"(lds_uniform) : "memory");
}

__device__ __forceinline__ bool wait_flag(const uint32_t* flag, uint32_t call, uint32_t* tmo) {
  if (ld_sys(flag) >= call) return true;
  unsigned long long t0 = __builtin_amdgcn_s_memrealtime();
  while (ld_sys(flag) < call) {
    if (ld_agent(tmo) == call || __builtin_amdgcn_s_memrealtime() - t0 > DEADLINE_TICKS) {
      __hip_atomic_store(tmo, call, __ATOMIC_RELAXED, __HIP_MEMORY_SCOPE_AGENT);
      return false;
    }
    __builtin_amdgcn_s_sleep(2);
  }
  return true;
}

__global__ __launch_bounds__(NTHREADS, 1) void fused_gemm_a2a(Params p) {
  __shared__ __attribute__((aligned(128))) uint8_t smem[NSTAGE * STAGE_BYTES];

  const int tid  = threadIdx.x;
  const int wid  = tid >> 6;
  const int lane = tid & 63;
  const int bid  = blockIdx.x;
  const int xcd  = bid & 7;
  const int idx  = bid >> 3;
  const int mt   = idx >> 2;
  const int nt   = xcd * 4 + (idx & 3);
  const int m0   = mt * BM;
  const int n0   = nt * BN;
  const unsigned dst = (unsigned)xcd;

  uint32_t* myws = p.ws[p.rank];
  STAMP(0);
  const uint32_t call = ld_agent(myws + W_CALLS) + 1u;

  const int lr = lane >> 3, lj = lane & 7;
  const uint8_t* a_src[2];
#pragma unroll
  for (int i = 0; i < 2; ++i) {
    int r = (wid * 2 + i) * 8 + lr;
    int kc = lj ^ ((r >> 1) & 7);
    a_src[i] = p.x + (size_t)(m0 + r) * K_DIM + kc * 16;
  }
  const uint8_t* b_src[4];
#pragma unroll
  for (int i = 0; i < 4; ++i) {
    int k = (wid * 4 + i) * 8 + lr;
    int nc = lj ^ (((k >> 1) & 3) | (((k >> 4) & 1) << 2));
    b_src[i] = p.w + (size_t)k * N_DIM + n0 + nc * 16;
  }

  const uint32_t lds0 = (uint32_t)(size_t)((__attribute__((address_space(3))) uint8_t*)smem);
  const uint32_t lds_a = __builtin_amdgcn_readfirstlane(lds0 + wid * 2048);
  const uint32_t lds_b = __builtin_amdgcn_readfirstlane(lds0 + A_BYTES + wid * 4096);
  auto issue = [&](int kt, int st) {
    const uint32_t sb = st * STAGE_BYTES;
#pragma unroll
    for (int i = 0; i < 2; ++i)
      glds16(a_src[i] + (size_t)kt * BK, lds_a + sb + i * 1024);
#pragma unroll
    for (int i = 0; i < 4; ++i)
      glds16(b_src[i] + (size_t)kt * BK * N_DIM, lds_b + sb + i * 1024);
  };

  const int g = lane >> 4, li = lane & 15;
  const int a_off0 = 128 * li + 16 * ((g ^ (li >> 1)) & 7);
  const int a_off1 = 128 * li + 16 * (((g ^ (li >> 1)) ^ 4) & 7);
  const int q = li >> 1, pp = li & 1;
  const int fB = ((q >> 1) & 3) | ((g & 1) << 2);
  int b_off[2];
#pragma unroll
  for (int j = 0; j < 2; ++j)
    b_off[j] = A_BYTES + 128 * (16 * g + q) + 8 * pp + 16 * ((wid * 2 + j) ^ fB);

  v4f acc[4][2];
#pragma unroll
  for (int i = 0; i < 4; ++i)
#pragma unroll
    for (int j = 0; j < 2; ++j) acc[i][j] = (v4f){0.f, 0.f, 0.f, 0.f};

  auto compute = [&](int st) {
    const uint8_t* sb = smem + st * STAGE_BYTES;
    v8i wf[2];
#pragma unroll
    for (int j = 0; j < 2; ++j) {
#pragma unroll
      for (int t = 0; t < 4; ++t) {
        v2i r = __builtin_amdgcn_ds_read_tr8_b64_v2i32(
            (v2i __attribute__((address_space(3)))*)(sb + b_off[j] + 8192 * (t >> 1) + 1024 * (t & 1)));
        wf[j][2 * t] = r.x;
        wf[j][2 * t + 1] = r.y;
      }
    }
#pragma unroll
    for (int mb = 0; mb < 4; ++mb) {
      v4i lo = *(const v4i*)(sb + 2048 * mb + a_off0);
      v4i hi = *(const v4i*)(sb + 2048 * mb + a_off1);
      v8i xf = {lo[0], lo[1], lo[2], lo[3], hi[0], hi[1], hi[2], hi[3]};
#pragma unroll
      for (int j = 0; j < 2; ++j)
        acc[mb][j] = __builtin_amdgcn_mfma_scale_f32_16x16x128_f8f6f4(wf[j], xf, acc[mb][j], 1, 0, 0, 0, 0, 0);
    }
  };

#pragma unroll
  for (int s = 0; s < NSTAGE - 1; ++s) issue(s, s);

  if (bid == 0 && wid == 0 && lane < NRANKS && lane != p.rank)
    __hip_atomic_store(p.ws[lane] + W_READY + 32 * p.rank, call, __ATOMIC_RELAXED, __HIP_MEMORY_SCOPE_SYSTEM);

  int st_c = 0;
  int st_i = NSTAGE - 1;
  for (int kt = 0; kt <= NT - NSTAGE; ++kt) {
    WAIT_VM(12);
    __builtin_amdgcn_s_barrier();
    issue(kt + NSTAGE - 1, st_i);
    compute(st_c);
    st_c = (st_c + 1) & (NSTAGE - 1);
    st_i = (st_i + 1) & (NSTAGE - 1);
  }
  WAIT_VM(12);
  __builtin_amdgcn_s_barrier();
  compute(st_c);
  st_c = (st_c + 1) & (NSTAGE - 1);
  WAIT_VM(6);
  __builtin_amdgcn_s_barrier();
  compute(st_c);
  st_c = (st_c + 1) & (NSTAGE - 1);
  WAIT_VM(0);
  __builtin_amdgcn_s_barrier();
  compute(st_c);

  STAMP(1);
  const float scale = p.sx[0] * p.sw[0];
  bool ok = true;
  if (dst != (unsigned)p.rank) ok = wait_flag(myws + W_READY + 32 * dst, call, myws + W_TMO);
  STAMP(2);
  float* obase = p.out[dst] + (size_t)(p.rank * M_LOC + m0) * N_LOC + (n0 - (int)dst * N_LOC) + wid * 32 + 4 * g;
#pragma unroll
  for (int mb = 0; mb < 4; ++mb) {
#pragma unroll
    for (int j = 0; j < 2; ++j) {
      v4f o;
#pragma unroll
      for (int r = 0; r < 4; ++r) {
        float y = acc[mb][j][r] * scale;
        float yc = fminf(fmaxf(y, -60.f), 60.f);
        o[r] = ok ? y / (1.f + __expf(-yc)) : __builtin_nanf("");
      }
      *(v4f*)(obase + (size_t)(16 * mb + li) * N_LOC + 16 * j) = o;
    }
  }

  STAMP(3);
  WAIT_VM(0);
  __syncthreads();
  STAMP(4);
  if (tid == 0) {
    uint32_t t = __hip_atomic_fetch_add(myws + W_TICKET, 1u, __ATOMIC_RELEASE, __HIP_MEMORY_SCOPE_AGENT);
    STAMP(5);
    if ((t % GRID) == GRID - 1) {
      __builtin_amdgcn_fence(__ATOMIC_ACQUIRE, "agent");
      __builtin_amdgcn_fence(__ATOMIC_RELEASE, "");
      WAIT_VM(0);
#pragma unroll
      for (int r = 0; r < NRANKS; ++r)
        if (r != p.rank)
          __hip_atomic_store(p.ws[r] + W_DONE + 32 * p.rank, call, __ATOMIC_RELAXED, __HIP_MEMORY_SCOPE_SYSTEM);
      bool all_ok = true;
      for (int r = 0; r < NRANKS; ++r)
        if (r != p.rank) all_ok &= wait_flag(myws + W_DONE + 32 * r, call, myws + W_TMO);
      __builtin_amdgcn_fence(__ATOMIC_ACQUIRE, "");
      if (!all_ok) p.out[p.rank][(size_t)p.rank * M_LOC * N_LOC] = __builtin_nanf("");
      __hip_atomic_store(myws + W_CALLS, call, __ATOMIC_RELAXED, __HIP_MEMORY_SCOPE_AGENT);
      STAMP(6);
    }
  }
}

}

extern "C" void kernel_launch(int rank, int nranks, ncclComm_t comm,
                              hipStream_t stream,
                              void* const* d_in, const int* in_sizes, int n_in,
                              void* d_out, int out_size,
                              const int* in_dtypes, int out_dtype,
                              void* const* d_out_all,
                              void* const* d_ws_all, size_t ws_bytes) {
  (void)comm; (void)in_sizes; (void)n_in; (void)out_size; (void)in_dtypes; (void)out_dtype; (void)ws_bytes;
  if (nranks != NRANKS) { fprintf(stderr, "expected %d ranks\n", NRANKS); return; }
  Params p;
  p.x  = (const uint8_t*)d_in[0];
  p.w  = (const uint8_t*)d_in[1];
  p.sx = (const float*)d_in[2];
  p.sw = (const float*)d_in[3];
  for (int r = 0; r < NRANKS; ++r) {
    p.out[r] = (float*)d_out_all[r];
    p.ws[r]  = (uint32_t*)d_ws_all[r];
  }
  p.out[rank] = (float*)d_out;
  p.rank = rank;
  hipLaunchKernelGGL(fused_gemm_a2a, dim3(GRID), dim3(NTHREADS), 0, stream, p);
  HIP_CHECK(hipGetLastError());
}
```

```cpp
#include <hip/hip_runtime.h>
#include <rccl/rccl.h>
#include <stdint.h>
#include <cstdio>

#define HIP_CHECK(expr) do { hipError_t _e = (expr); if (_e != hipSuccess) { \
    fprintf(stderr, "HIP error %s at %s:%d\n", hipGetErrorString(_e), __FILE__, __LINE__); \
    return; } } while (0)

namespace {

constexpr int NRANKS = 8;
constexpr int M_LOC  = 512;
constexpr int K_DIM  = 4096;
constexpr int N_DIM  = 4096;
constexpr int N_LOC  = 512;

constexpr int BM = 64, BN = 128, BK = 128;
#define NSTAGE_DEF 6
constexpr int NSTAGE = NSTAGE_DEF;
constexpr int A_BYTES = BM * BK;
constexpr int B_BYTES = BK * BN;
constexpr int STAGE_BYTES = A_BYTES + B_BYTES;
constexpr int NT = K_DIM / BK;
#define NLW 4
constexpr int NLOAD = NLW;
constexpr int PPW = 24 / NLOAD;
constexpr int NTHREADS = 256 + 64 * NLOAD;
constexpr int GRID = (M_LOC / BM) * (N_DIM / BN);

constexpr int W_CALLS  = 0;
constexpr int W_TICKET = 32;
constexpr int W_TMO    = 64;
constexpr int W_READY  = 256;
constexpr int W_DONE   = 512;
constexpr unsigned long long DEADLINE_TICKS = 100000000ull;

typedef int   v2i __attribute__((ext_vector_type(2)));
typedef int   v4i __attribute__((ext_vector_type(4)));
typedef int   v8i __attribute__((ext_vector_type(8)));
typedef float v4f __attribute__((ext_vector_type(4)));

struct Params {
  const uint8_t* x;
  const uint8_t* w;
  const float* sx;
  const float* sw;
  float* out[NRANKS];
  uint32_t* ws[NRANKS];
  int rank;
};

#define STAMP(i) do {} while (0)
#define WAIT_VM(n) asm volatile("s_waitcnt vmcnt(" #n ")" ::: "memory")

__device__ __forceinline__ uint32_t ld_sys(const uint32_t* p) {
  return __hip_atomic_load(p, __ATOMIC_RELAXED, __HIP_MEMORY_SCOPE_SYSTEM);
}
__device__ __forceinline__ uint32_t ld_agent(const uint32_t* p) {
  return __hip_atomic_load(p, __ATOMIC_RELAXED, __HIP_MEMORY_SCOPE_AGENT);
}

__device__ __forceinline__ void glds16(const uint8_t* src, uint32_t lds_uniform) {
  asm volatile("s_mov_b32 m0, %1\n\ts_nop 0\n\tglobal_load_lds_dwordx4 %0, off"
               :: "v"(src), "s"(lds_uniform) : "memory");
}

__device__ __forceinline__ void store_sys(float* ptr, v4f v) {
  asm volatile("global_store_dwordx4 %0, %1, off sc0 sc1\n\ts_nop 1" :: "v"(ptr), "v"(v) : "memory");
}

__device__ __forceinline__ bool wait_flag(const uint32_t* flag, uint32_t call, uint32_t* tmo) {
  if (ld_sys(flag) >= call) return true;
  unsigned long long t0 = __builtin_amdgcn_s_memrealtime();
  while (ld_sys(flag) < call) {
    if (ld_agent(tmo) == call || __builtin_amdgcn_s_memrealtime() - t0 > DEADLINE_TICKS) {
      __hip_atomic_store(tmo, call, __ATOMIC_RELAXED, __HIP_MEMORY_SCOPE_AGENT);
      return false;
    }
    __builtin_amdgcn_s_sleep(2);
  }
  return true;
}

__global__ __launch_bounds__(NTHREADS, 1) void fused_gemm_a2a(Params p) {
  __shared__ __attribute__((aligned(128))) uint8_t smem[NSTAGE * STAGE_BYTES];

  const int tid  = threadIdx.x;
  const int wid  = __builtin_amdgcn_readfirstlane(tid >> 6);
  const bool loader = wid >= 4;
  const int cw = wid & 3;
  const int lw = wid - 4;
  const int lane = tid & 63;
  const int bid  = blockIdx.x;
  const int xcd  = bid & 7;
  const int idx  = bid >> 3;
  const int mt   = idx >> 2;
  const int nt   = xcd * 4 + (idx & 3);
  const int m0   = mt * BM;
  const int n0   = nt * BN;
  const unsigned dst = (unsigned)xcd;

  uint32_t* myws = p.ws[p.rank];
  STAMP(0);
  const uint32_t call = ld_agent(myws + W_CALLS) + 1u;

  const int lr = lane >> 3, lj = lane & 7;
  const uint8_t* src[PPW];
#pragma unroll
  for (int i = 0; i < PPW; ++i) {
    const int piece = lw * PPW + i;
    if (piece < 8) {
      int r = piece * 8 + lr;
      int kc = lj ^ ((r >> 1) & 7);
      src[i] = p.x + (size_t)(m0 + r) * K_DIM + kc * 16;
    } else {
      int k = (piece - 8) * 8 + lr;
      int nc = lj ^ (((k >> 1) & 3) | (((k >> 4) & 1) << 2));
      src[i] = p.w + (size_t)k * N_DIM + n0 + nc * 16;
    }
  }
  const uint32_t lds0 = (uint32_t)(size_t)((__attribute__((address_space(3))) uint8_t*)smem);
  const uint32_t lds_w = __builtin_amdgcn_readfirstlane(lds0 + lw * PPW * 1024);
  auto issue = [&](int kt, int st) {
    const uint32_t sb = st * STAGE_BYTES;
#pragma unroll
    for (int i = 0; i < PPW; ++i) {
      const int piece = lw * PPW + i;
      const size_t adv = (piece < 8) ? (size_t)kt * BK : (size_t)kt * BK * N_DIM;
      glds16(src[i] + adv, lds_w + sb + i * 1024);
    }
  };

  const int g = lane >> 4, li = lane & 15;
  const int a_off0 = 128 * li + 16 * ((g ^ (li >> 1)) & 7);
  const int a_off1 = 128 * li + 16 * (((g ^ (li >> 1)) ^ 4) & 7);
  const int q = li >> 1, pp = li & 1;
  const int fB = ((q >> 1) & 3) | ((g & 1) << 2);
  int b_off[2];
#pragma unroll
  for (int j = 0; j < 2; ++j)
    b_off[j] = A_BYTES + 128 * (16 * g + q) + 8 * pp + 16 * ((cw * 2 + j) ^ fB);

  v4f acc[4][2];
#pragma unroll
  for (int i = 0; i < 4; ++i)
#pragma unroll
    for (int j = 0; j < 2; ++j) acc[i][j] = (v4f){0.f, 0.f, 0.f, 0.f};

  struct Frags { v8i wf[2]; v8i xf[4]; };
  auto load_frags = [&](Frags& fr, int st) {
    const uint8_t* sb = smem + st * STAGE_BYTES;
#pragma unroll
    for (int j = 0; j < 2; ++j) {
#pragma unroll
      for (int t = 0; t < 4; ++t) {
        v2i r = __builtin_amdgcn_ds_read_tr8_b64_v2i32(
            (v2i __attribute__((address_space(3)))*)(sb + b_off[j] + 8192 * (t >> 1) + 1024 * (t & 1)));
        fr.wf[j][2 * t] = r.x;
        fr.wf[j][2 * t + 1] = r.y;
      }
    }
#pragma unroll
    for (int mb = 0; mb < 4; ++mb) {
      v4i lo = *(const v4i*)(sb + 2048 * mb + a_off0);
      v4i hi = *(const v4i*)(sb + 2048 * mb + a_off1);
      fr.xf[mb] = (v8i){lo[0], lo[1], lo[2], lo[3], hi[0], hi[1], hi[2], hi[3]};
    }
  };
  auto mma = [&](const Frags& fr) {
#pragma unroll
    for (int mb = 0; mb < 4; ++mb)
#pragma unroll
      for (int j = 0; j < 2; ++j)
        acc[mb][j] = __builtin_amdgcn_mfma_scale_f32_16x16x128_f8f6f4(fr.wf[j], fr.xf[mb], acc[mb][j], 1, 0, 0, 0, 0, 0);
  };

  if (loader) {
#pragma unroll
    for (int s = 0; s < NSTAGE; ++s) issue(s, s);
    asm volatile("s_waitcnt vmcnt(%0)" :: "i"(PPW * (NSTAGE - 1)) : "memory");
    __builtin_amdgcn_s_barrier();
    int st_i = 0;
    for (int kt = 0; kt < NT - NSTAGE; ++kt) {
      asm volatile("s_waitcnt vmcnt(%0)" :: "i"(PPW * (NSTAGE - 2)) : "memory");
      __builtin_amdgcn_s_barrier();
      issue(kt + NSTAGE, st_i);
      st_i = (st_i + 1 == NSTAGE) ? 0 : st_i + 1;
    }
    asm volatile("s_waitcnt vmcnt(0)" ::: "memory");
#pragma unroll
    for (int r = 0; r < NSTAGE; ++r) __builtin_amdgcn_s_barrier();
  } else {
    if (bid == 0 && wid == 0 && lane < NRANKS && lane != p.rank)
      __hip_atomic_store(p.ws[lane] + W_READY + 32 * p.rank, call, __ATOMIC_RELAXED, __HIP_MEMORY_SCOPE_SYSTEM);
    Frags f0, f1;
    __builtin_amdgcn_s_barrier();
    load_frags(f0, 0);
    int st_n = 1;
    for (int kt = 0; kt < NT; kt += 2) {
      asm volatile("s_waitcnt lgkmcnt(0)" ::: "memory");
      __builtin_amdgcn_s_barrier();

      load_frags(f1, st_n);
      __builtin_amdgcn_sched_barrier(0);
      mma(f0);
      __builtin_amdgcn_sched_barrier(0);
      st_n = (st_n + 1 == NSTAGE) ? 0 : st_n + 1;
      asm volatile("s_waitcnt lgkmcnt(0)" ::: "memory");
      __builtin_amdgcn_s_barrier();
      if (kt + 2 < NT) load_frags(f0, st_n);
      __builtin_amdgcn_sched_barrier(0);
      mma(f1);
      __builtin_amdgcn_sched_barrier(0);
      st_n = (st_n + 1 == NSTAGE) ? 0 : st_n + 1;
    }
  }

  STAMP(1);
  if (!loader) {
    const float scale = p.sx[0] * p.sw[0];
    bool ok = true;
    if (dst != (unsigned)p.rank) ok = wait_flag(myws + W_READY + 32 * dst, call, myws + W_TMO);
    STAMP(2);
    float* obase = p.out[dst] + (size_t)(p.rank * M_LOC + m0) * N_LOC + (n0 - (int)dst * N_LOC) + cw * 32 + 4 * g;
#pragma unroll
    for (int mb = 0; mb < 4; ++mb) {
#pragma unroll
      for (int j = 0; j < 2; ++j) {
        v4f o;
#pragma unroll
        for (int r = 0; r < 4; ++r) {
          float y = acc[mb][j][r] * scale;
          float yc = fminf(fmaxf(y, -60.f), 60.f);
          o[r] = ok ? y * __builtin_amdgcn_rcpf(1.f + __expf(-yc)) : __builtin_nanf("");
        }
        if (dst != (unsigned)p.rank) store_sys(obase + (size_t)(16 * mb + li) * N_LOC + 16 * j, o);
        else *(v4f*)(obase + (size_t)(16 * mb + li) * N_LOC + 16 * j) = o;
      }
    }
  }

  STAMP(3);
  WAIT_VM(0);
  __syncthreads();
  STAMP(4);
  if (wid == 0) {
    uint32_t t = 0;
    if (lane == 0) {
      if (dst != (unsigned)p.rank) {
        __builtin_amdgcn_fence(__ATOMIC_RELEASE, "");
        WAIT_VM(0);
        __hip_atomic_store(p.ws[dst] + W_DONE + 32 * p.rank + idx, call, __ATOMIC_RELAXED, __HIP_MEMORY_SCOPE_SYSTEM);
      }
      t = __hip_atomic_fetch_add(myws + W_TICKET, 1u, __ATOMIC_RELAXED, __HIP_MEMORY_SCOPE_AGENT);
      STAMP(5);
    }
    t = __builtin_amdgcn_readfirstlane(t);
    if ((t % GRID) == GRID - 1) {
      if (lane == 0) __hip_atomic_store(myws + W_CALLS, call, __ATOMIC_RELAXED, __HIP_MEMORY_SCOPE_AGENT);
      const uint32_t* fl = myws + W_DONE + 4 * lane;
      const bool mine = (lane >> 3) == p.rank;
      unsigned long long t0 = __builtin_amdgcn_s_memrealtime();
      bool ok = true;
      for (;;) {
        uint32_t a = ld_sys(fl), b = ld_sys(fl + 1), c = ld_sys(fl + 2), d = ld_sys(fl + 3);
        bool done = mine || (a >= call && b >= call && c >= call && d >= call);
        if (__all(done)) break;
        if (ld_agent(myws + W_TMO) == call || __builtin_amdgcn_s_memrealtime() - t0 > DEADLINE_TICKS) { ok = false; break; }
        __builtin_amdgcn_s_sleep(1);
      }
      if (!ok && lane == 0) {
        __hip_atomic_store(myws + W_TMO, call, __ATOMIC_RELAXED, __HIP_MEMORY_SCOPE_AGENT);
        p.out[p.rank][(size_t)p.rank * M_LOC * N_LOC] = __builtin_nanf("");
      }
      STAMP(6);
    }
  }
}

}

extern "C" void kernel_launch(int rank, int nranks, ncclComm_t comm,
                              hipStream_t stream,
                              void* const* d_in, const int* in_sizes, int n_in,
                              void* d_out, int out_size,
                              const int* in_dtypes, int out_dtype,
                              void* const* d_out_all,
                              void* const* d_ws_all, size_t ws_bytes) {
  (void)comm; (void)in_sizes; (void)n_in; (void)out_size; (void)in_dtypes; (void)out_dtype; (void)ws_bytes;
  if (nranks != NRANKS) { fprintf(stderr, "expected %d ranks\n", NRANKS); return; }
  Params p;
  p.x  = (const uint8_t*)d_in[0];
  p.w  = (const uint8_t*)d_in[1];
  p.sx = (const float*)d_in[2];
  p.sw = (const float*)d_in[3];
  for (int r = 0; r < NRANKS; ++r) {
    p.out[r] = (float*)d_out_all[r];
    p.ws[r]  = (uint32_t*)d_ws_all[r];
  }
  p.out[rank] = (float*)d_out;
  p.rank = rank;
  hipLaunchKernelGGL(fused_gemm_a2a, dim3(GRID), dim3(NTHREADS), 0, stream, p);
  HIP_CHECK(hipGetLastError());
}
```
